```python
import numpy as np

kernel_name = "gpu_gemm_a2a_m4096_k4096_n16384_e4m3e5m2_dqsilu"


ABSMAX_RTOL = 0.05

COMPUTE_DTYPE = "float32"
EPILOGUE = "dequant_silu"
IN_QUANT = None
OUT_QUANT = None
IN_FP8 = "e4m3e5m2"
IN_DTYPES = {"x": "e4m3", "w_mat": "e5m2",
             "scale_x": "float32", "scale_w": "float32"}
WORLD_SIZE = 8

def _snap_fp8(x, mant_bits, max_abs, min_normal):
    a = np.abs(x.astype(np.float64))
    _m, _e = np.frexp(a)
    q = np.where(a >= min_normal, np.ldexp(1.0, _e - 1 - mant_bits),
                 min_normal * 2.0 ** -mant_bits)
    snapped = np.minimum(np.round(a / q) * q, max_abs)
    return (np.sign(x) * snapped).astype(np.float32)

def setup_inputs():

    x = _snap_fp8(np.random.randn(4096, 4096).astype(np.float32), 3, 448.0, 2.0 ** -6)
    w_mat = _snap_fp8(np.random.randn(4096, 16384).astype(np.float32), 2, 57344.0, 2.0 ** -14)
    scale_x = (0.02 * (1.0 + np.random.rand(1))).astype(np.float32)
    scale_w = (0.02 * (1.0 + np.random.rand(1))).astype(np.float32)
    return {"x": x, "w_mat": w_mat, "scale_x": scale_x, "scale_w": scale_w}

def reference(x, w_mat, scale_x, scale_w):
    acc = (x @ w_mat).astype(np.float32)
    y = (acc * (scale_x * scale_w)).astype(np.float32)
    return (y / (1.0 + np.exp(-np.clip(y, -60.0, 60.0)))).astype(np.float32)
```

```cpp
#include <hip/hip_runtime.h>
#include <rccl/rccl.h>
#include <cstdio>

#define HIP_CHECK(expr) do { hipError_t _e = (expr); if (_e != hipSuccess) { \
    fprintf(stderr, "HIP error %s at %s:%d\n", hipGetErrorString(_e), __FILE__, __LINE__); \
    return; } } while (0)

namespace {
typedef unsigned u32x4 __attribute__((ext_vector_type(4)));
typedef int i32x2 __attribute__((ext_vector_type(2)));
typedef int i32x4 __attribute__((ext_vector_type(4)));
typedef int i32x8 __attribute__((ext_vector_type(8)));
typedef float f32x4 __attribute__((ext_vector_type(4)));
#define LAS __attribute__((address_space(3)))
#define GAS __attribute__((address_space(1)))

constexpr int NOWN = 5;
constexpr int NR = 8;
constexpr int MS = 512;
constexpr int KD = 4096;
constexpr int NG = 16384;
constexpr int NS = 2048;
constexpr int BN = 256, BK = 128;
constexpr int NKS = KD / BK;
constexpr int MT_BYTES = 16 * BK;
constexpr int A_BYTES = 10 * MT_BYTES;
constexpr int B_BYTES = BK * BN;
constexpr int STAGE = A_BYTES + B_BYTES;
constexpr int NSTAGE = 3;
constexpr int SYNC_OFF = NSTAGE * STAGE;
constexpr int LDS_BYTES = SYNC_OFF + 256;
constexpr int NCW = 32;
constexpr int RS = (256 - NCW) / 16;
constexpr size_t SLOT_BYTES = (size_t)MS * KD;
constexpr size_t XS_BYTES = NR * SLOT_BYTES;
constexpr size_t FLAG_OFF = 2 * XS_BYTES;
constexpr size_t CTR_OFF = FLAG_OFF + NR * 128;
constexpr size_t TMO_OFF = CTR_OFF + 256 * 8 * 4;
constexpr size_t WS_NEED = TMO_OFF + 128;
constexpr unsigned long long DEADLINE_TICKS = 20000000ull;

struct Params {
    const unsigned char* x;
    const unsigned char* w;
    const float* sx;
    const float* sw;
    float* out;
    unsigned char* myws;
    unsigned char* dws[8];
    int rank;
};

__device__ __forceinline__ unsigned lds_ld(LAS unsigned char* lds, int off) {
    return __hip_atomic_load((LAS unsigned*)(lds + off), __ATOMIC_RELAXED, __HIP_MEMORY_SCOPE_WORKGROUP);
}
__device__ __forceinline__ void lds_st(LAS unsigned char* lds, int off, unsigned v) {
    __hip_atomic_store((LAS unsigned*)(lds + off), v, __ATOMIC_RELAXED, __HIP_MEMORY_SCOPE_WORKGROUP);
}
constexpr int FULL_OFF = SYNC_OFF, FREE_OFF = SYNC_OFF + 64, BAD_OFF = SYNC_OFF + 128, HAND_OFF = SYNC_OFF + 192;

__device__ __forceinline__ bool test_min4(LAS unsigned char* lds, int off, unsigned need) {
    u32x4 f = *(volatile LAS u32x4*)(lds + off);
    unsigned m = min(min(f[0], f[1]), min(f[2], f[3]));
    return (unsigned)__builtin_amdgcn_readfirstlane(m) >= need;
}
__device__ __forceinline__ void wait_min4(LAS unsigned char* lds, int off, unsigned need) {
    while (!test_min4(lds, off, need)) __builtin_amdgcn_s_sleep(1);
    asm volatile("" ::: "memory");
}

constexpr int COMM_U = 2 * NKS;
constexpr int COMM_P = 4;
constexpr int COMM_W = 3;
constexpr int comm_wait(int u) {
    int n = (u - COMM_W + COMM_P < COMM_U) ? 1 : 0;
    for (int j = u - COMM_W + 1; j < u; ++j) n += 1 + ((j + COMM_P < COMM_U) ? 1 : 0);
    return n;
}
struct CommCtx {
    const unsigned char* src;
    __amdgpu_buffer_rsrc_t rs;
    unsigned doff;
    unsigned soff;
    unsigned* flag;
    unsigned val;
    bool pub;
    u32x4 v[COMM_P];
};
constexpr int comm_soff(int u) { return (u / 2) * BK + (u % 2) * 64; }
template <int J, int UREF> __device__ __forceinline__ void comm_step(CommCtx& c) {
    if constexpr (UREF >= COMM_W) {
        asm volatile("s_waitcnt vmcnt(%0)" :: "n"(comm_wait(UREF)) : "memory");
        constexpr int acked = J - COMM_W + 1;
        if constexpr ((acked & 1) == 0) {
            __builtin_amdgcn_s_barrier();
            if (c.pub) __hip_atomic_store(c.flag, c.val + (unsigned)(acked / 2), __ATOMIC_RELAXED, __HIP_MEMORY_SCOPE_SYSTEM);
        }
    }
    __builtin_amdgcn_raw_buffer_store_b128(c.v[J], c.rs, c.doff, c.soff + (J / 2) * (32 * MT_BYTES) + (J % 2) * 1024, 17  );
    if constexpr (UREF + COMM_P < COMM_U) c.v[J] = *(const u32x4*)(c.src + comm_soff(J + COMM_P));
    asm volatile("" ::: "memory");
}
template <int U0> __device__ __forceinline__ void comm_trip(CommCtx& c) {
    comm_step<0, U0>(c); comm_step<1, U0 + 1>(c); comm_step<2, U0 + 2>(c); comm_step<3, U0 + 3>(c);
    c.src += 2 * BK; c.soff += 2 * 32 * MT_BYTES; c.val += 2;
}

__device__ __forceinline__ f32x4 silu4(f32x4 a, float scale, float nsl) {
    f32x4 o;
#pragma unroll
    for (int r = 0; r < 4; ++r) {
        const float t = __builtin_amdgcn_fmed3f(a[r] * nsl, -86.5617024533378f, 86.5617024533378f);
        o[r] = (a[r] * scale) * __builtin_amdgcn_rcpf(1.0f + __builtin_amdgcn_exp2f(t));
    }
    return o;
}
#define OUT_ST(ptr, v) asm volatile("global_store_dwordx4 %0, %1, off sc1\n\ts_nop 1" :: "v"(ptr), "v"(v) : "memory")
__device__ __forceinline__ int ho_off(int h, int wn) { return (h < 3 ? h * 16384 : 2 * STAGE + (h - 3) * 16384) + wn * 4096; }
constexpr int PRIV_OFF = 2 * STAGE + 32768;

__global__ void __launch_bounds__(512, 2) fused_kernel(Params p) {
    __shared__ __attribute__((aligned(1024))) unsigned char lds_raw[LDS_BYTES];
    LAS unsigned char* lds = (LAS unsigned char*)lds_raw;
    const int tid = threadIdx.x, lane = tid & 63;
    const int wid = __builtin_amdgcn_readfirstlane(tid >> 6);
    const int b = blockIdx.x;
    unsigned char* myws = p.myws;
    unsigned* ctr = (unsigned*)(myws + CTR_OFF) + b * 8 + wid;

    if (b < NCW) {
        const int x = b & 7;
        unsigned char* dws = p.dws[(x >> 2) * 4 + (b >> 3)];
        const unsigned char* px = p.x;
        const int rank = p.rank;
        asm volatile("" :: "s"(dws), "s"(px), "s"(rank), "s"(myws));
        const int mt0 = 8 * (x & 3), mt = mt0 + wid;
        CommCtx c;
        unsigned call = __hip_atomic_load(ctr, __ATOMIC_RELAXED, __HIP_MEMORY_SCOPE_AGENT) + 1u;
        c.src = px + (size_t)(16 * mt + (lane & 15)) * KD + 16 * (lane >> 4);
#pragma unroll
        for (int i = 0; i < COMM_P; ++i) c.v[i] = *(const u32x4*)(c.src + comm_soff(i));
        if (dws == nullptr) return;
        if (lane == 0) __hip_atomic_store(ctr, call, __ATOMIC_RELAXED, __HIP_MEMORY_SCOPE_AGENT);
        call = __builtin_amdgcn_readfirstlane(call);
        c.rs = __builtin_amdgcn_make_buffer_rsrc(dws + (size_t)(call & 1u) * XS_BYTES + (size_t)rank * SLOT_BYTES, 0, (int)SLOT_BYTES, 0x00020000);
        c.doff = (unsigned)(mt * MT_BYTES + lane * 16);
        c.pub = tid < 8;
        c.flag = (unsigned*)(dws + FLAG_OFF + (size_t)rank * 128) + mt0 + (lane & 7);
        c.val = call * 64u;
        c.soff = 0;
        comm_trip<0>(c);
#pragma unroll 1
        for (int t = 1; t < COMM_U / COMM_P - 1; ++t) comm_trip<COMM_P>(c);
        comm_trip<COMM_U - COMM_P>(c);
        asm volatile("s_waitcnt vmcnt(2)" ::: "memory");
        __builtin_amdgcn_s_barrier();
        if (c.pub) __hip_atomic_store(c.flag, c.val - 1u, __ATOMIC_RELAXED, __HIP_MEMORY_SCOPE_SYSTEM);
        asm volatile("s_waitcnt vmcnt(0)" ::: "memory");
        __builtin_amdgcn_s_barrier();
        if (c.pub) __hip_atomic_store(c.flag, c.val, __ATOMIC_RELAXED, __HIP_MEMORY_SCOPE_SYSTEM);
        return;
    }
    const int xg = b & 7, idx = (b - NCW) >> 3;
    const int hf = xg >> 2, cb = 2 * (xg & 3) + (idx & 1), rs = idx >> 1;
    const int p0 = 128 * rs / RS, nmt = 128 * (rs + 1) / RS - p0;
#define POS_SO(pos) ((pos) >> 4)
#define POS_MT(pos) (hf * 16 + ((pos) & 15))
    const bool ten = nmt == 10;

    if (tid < 64) lds_st(lds, SYNC_OFF + tid * 4, 0u);

    auto helper_epilogue = [&]() {
        const int wn = wid - 4, g = lane >> 4;
        float scale = p.sx[0] * p.sw[0];
        float* ocol = p.out + cb * BN + wn * 64 + 4 * (lane & 15);
        for (int mt = NOWN; mt < nmt; ++mt) {
            while ((unsigned)__builtin_amdgcn_readfirstlane(lds_ld(lds, HAND_OFF + 4 * wn)) < (unsigned)(mt - NOWN + 1)) __builtin_amdgcn_s_sleep(1);
            asm volatile("" ::: "memory");
            if (mt == NOWN && (lds_ld(lds, BAD_OFF) | lds_ld(lds, BAD_OFF + 4)) != 0u) scale = __builtin_nanf("");
            const float nsl = scale * -1.44269504088896341f;
            const int pos = p0 + mt;
            float* orow = ocol + (size_t)(((p.rank + POS_SO(pos)) & 7) * MS + 16 * POS_MT(pos) + g) * NS;
            LAS unsigned char* src = lds + ho_off(mt - NOWN, wn) + g * 256;
#pragma unroll
            for (int i = 0; i < 4; ++i) {
                const f32x4 o = silu4(*(const LAS f32x4*)(src + i * 1024 + (((lane & 15) ^ (4 * i + g)) << 4)), scale, nsl);
                OUT_ST(orow + (size_t)(4 * i) * NS, o);
            }
        }
    };

    if (wid == 4 || wid == 7) {
        const int lw = wid == 7 ? 1 : 0;
        unsigned call = __hip_atomic_load(ctr, __ATOMIC_RELAXED, __HIP_MEMORY_SCOPE_AGENT) + 1u;
        if (lane == 0) __hip_atomic_store(ctr, call, __ATOMIC_RELAXED, __HIP_MEMORY_SCOPE_AGENT);
        call = __builtin_amdgcn_readfirstlane(call);
        __builtin_amdgcn_s_barrier();
        const int n5 = lw == 0 ? 5 : nmt - 5;
        const unsigned char* ptr[5]; unsigned kst[5], hst[5];
#pragma unroll
        for (int i = 0; i < 5; ++i) {
            const int pos = p0 + 5 * lw + (i < n5 ? i : 0), so = POS_SO(pos), mt = POS_MT(pos);
            if (so == 0) { ptr[i] = p.x + (size_t)(16 * mt + (lane & 15)) * KD + 16 * (lane >> 4); kst[i] = BK; hst[i] = 64; }
            else { ptr[i] = myws + (size_t)(call & 1u) * XS_BYTES + (size_t)((p.rank + so) & 7) * SLOT_BYTES + mt * MT_BYTES + lane * 16; kst[i] = 32 * MT_BYTES; hst[i] = 1024; }
        }
        const int fpos = p0 + 5 * lw + (lane < n5 ? lane : 0), fso = POS_SO(fpos);
        const bool fremote = lane < n5 && fso != 0;
        const unsigned* flags = (const unsigned*)(myws + FLAG_OFF + (size_t)((p.rank + fso) & 7) * 128) + POS_MT(fpos);
        unsigned* tmo = (unsigned*)(myws + TMO_OFF);
        const unsigned base = call * 64u;
        unsigned avail = __builtin_amdgcn_ballot_w64(fremote) ? 0u : (unsigned)NKS;
        bool dead = false;
        int sb = lw * 5 * MT_BYTES;
        const int fo = FULL_OFF + 12 * lw;
        for (int ks = 0; ks < NKS; ++ks) {
            if (avail < (unsigned)(ks + 1) && !dead) {
                asm volatile("s_waitcnt vmcnt(0)" ::: "memory");
                if (lane == 0) lds_st(lds, fo, (unsigned)ks);
                const unsigned long long t0 = __builtin_amdgcn_s_memrealtime();
                for (unsigned it = 1;; ++it) {
                    unsigned f = 0xffffffffu;
                    if (fremote) f = __hip_atomic_load(flags, __ATOMIC_RELAXED, __HIP_MEMORY_SCOPE_SYSTEM);
                    unsigned m = __builtin_amdgcn_readlane(f, 0);
#pragma unroll
                    for (int i = 1; i < 5; ++i) m = min(m, (unsigned)__builtin_amdgcn_readlane(f, i));
                    if (m >= base + (unsigned)(ks + 1)) { avail = m - base; break; }
                    if ((it & 15u) == 0u &&
                        ((unsigned)__builtin_amdgcn_readfirstlane(__hip_atomic_load(tmo, __ATOMIC_RELAXED, __HIP_MEMORY_SCOPE_AGENT)) == call ||
                         __builtin_amdgcn_s_memrealtime() - t0 > DEADLINE_TICKS)) {
                        if (lane == 0) __hip_atomic_store(tmo, call, __ATOMIC_RELAXED, __HIP_MEMORY_SCOPE_AGENT);
                        dead = true; break;
                    }
                    __builtin_amdgcn_s_sleep(1);
                }
                asm volatile("" ::: "memory");
            }
            if (ks >= NSTAGE && !test_min4(lds, FREE_OFF, (unsigned)(ks - NSTAGE + 1))) {
                asm volatile("s_waitcnt vmcnt(0)" ::: "memory");
                if (lane == 0) lds_st(lds, fo, (unsigned)ks);
                wait_min4(lds, FREE_OFF, (unsigned)(ks - NSTAGE + 1));
            }
            asm volatile("" ::: "memory");
            if (!dead) {
#pragma unroll
                for (int i = 0; i < 5; ++i) {
                    const unsigned char* gp = ptr[i] + (size_t)ks * kst[i];
                    __builtin_amdgcn_global_load_lds((const GAS unsigned*)gp, (LAS unsigned*)(lds + sb + i * MT_BYTES), 16, 0, 17);
                    __builtin_amdgcn_global_load_lds((const GAS unsigned*)(gp + hst[i]), (LAS unsigned*)(lds + sb + i * MT_BYTES + 1024), 16, 0, 17);
                }
                asm volatile("s_waitcnt vmcnt(10)" ::: "memory");
            }
            if (lane == 0) lds_st(lds, fo, (unsigned)ks);
            sb += STAGE; if (sb >= NSTAGE * STAGE) sb -= NSTAGE * STAGE;
        }
        asm volatile("s_waitcnt vmcnt(0)" ::: "memory");
        dead = dead || ((unsigned)__builtin_amdgcn_readfirstlane(__hip_atomic_load(tmo, __ATOMIC_RELAXED, __HIP_MEMORY_SCOPE_AGENT)) == call);
        if (lane == 0) { lds_st(lds, BAD_OFF + 4 * lw, dead ? 1u : 0u); lds_st(lds, fo, (unsigned)NKS); }
        helper_epilogue();
        return;
    }
    if (wid >= 5) {
        const int j = wid - 5, r4 = lane >> 4, s = lane & 15;
        const unsigned char* wb = p.w + (size_t)(64 * j + r4) * NG + (size_t)p.rank * NS + cb * BN;
        unsigned voff[4];
#pragma unroll
        for (int v = 0; v < 4; ++v) voff[v] = (unsigned)((s ^ (r4 | ((v & 1) << 2) | ((v >> 1) << 3))) << 4);
        __builtin_amdgcn_s_barrier();
        int sb = A_BYTES + 64 * j * BN;
        for (int ks = 0; ks < NKS; ++ks) {
            if (ks >= NSTAGE && !test_min4(lds, FREE_OFF, (unsigned)(ks - NSTAGE + 1))) {
                asm volatile("s_waitcnt vmcnt(0)" ::: "memory");
                if (lane == 0) lds_st(lds, FULL_OFF + 4 + 4 * j, (unsigned)ks);
                wait_min4(lds, FREE_OFF, (unsigned)(ks - NSTAGE + 1));
            }
            asm volatile("" ::: "memory");
            const unsigned char* gp = wb + (size_t)ks * BK * NG;
#pragma unroll
            for (int i = 0; i < 16; ++i) {
                const int v = (i & 1) | (((i >> 2) & 1) << 1);
                __builtin_amdgcn_global_load_lds((const GAS unsigned*)(gp + (size_t)(4 * i) * NG + voff[v]), (LAS unsigned*)(lds + sb + i * 1024), 16, 0, 0);
            }
            asm volatile("s_waitcnt vmcnt(16)" ::: "memory");
            if (lane == 0) lds_st(lds, FULL_OFF + 4 + 4 * j, (unsigned)ks);
            sb += STAGE; if (sb >= NSTAGE * STAGE) sb -= NSTAGE * STAGE;
        }
        asm volatile("s_waitcnt vmcnt(0)" ::: "memory");
        if (lane == 0) lds_st(lds, FULL_OFF + 4 + 4 * j, (unsigned)NKS);
        helper_epilogue();
        return;
    }
    const int wn = wid, g = lane >> 4, qq = (lane & 15) >> 1, pp = lane & 1;
    const int f = qq | ((g & 1) << 3);
    int baddr[4];
#pragma unroll
    for (int nt = 0; nt < 4; ++nt) baddr[nt] = A_BYTES + (16 * g + qq) * BN + (((4 * wn + nt) ^ f) << 4) + 8 * pp;
    const int aaddr = lane * 16;
    f32x4 acc[10][4];
#pragma unroll
    for (int mt = 0; mt < 10; ++mt)
#pragma unroll
        for (int nt = 0; nt < 4; ++nt) acc[mt][nt] = (f32x4){0.f, 0.f, 0.f, 0.f};
    float scale = p.sx[0] * p.sw[0];
    __builtin_amdgcn_s_barrier();
    int sb = 0;
#define RD_A(base, mt) ({ i32x4 a0_ = *(const LAS i32x4*)(lds + (base) + aaddr + (mt) * 2048); i32x4 a1_ = *(const LAS i32x4*)(lds + (base) + aaddr + (mt) * 2048 + 1024); \
                    (i32x8){a0_[0], a0_[1], a0_[2], a0_[3], a1_[0], a1_[1], a1_[2], a1_[3]}; })
#define RD_B(base, nt) ({ LAS unsigned char* bp_ = lds + (base) + baddr[nt]; \
                    i32x2 r0_ = __builtin_amdgcn_ds_read_tr8_b64_v2i32((LAS i32x2*)(bp_)); \
                    i32x2 r1_ = __builtin_amdgcn_ds_read_tr8_b64_v2i32((LAS i32x2*)(bp_ + 2048)); \
                    i32x2 r2_ = __builtin_amdgcn_ds_read_tr8_b64_v2i32((LAS i32x2*)(bp_ + 16384)); \
                    i32x2 r3_ = __builtin_amdgcn_ds_read_tr8_b64_v2i32((LAS i32x2*)(bp_ + 16384 + 2048)); \
                    (i32x8){r0_[0], r0_[1], r1_[0], r1_[1], r2_[0], r2_[1], r3_[0], r3_[1]}; })
#define MM1(mt, nt, a) acc[mt][nt] = __builtin_amdgcn_mfma_scale_f32_16x16x128_f8f6f4(bf[nt], a, acc[mt][nt], 1  , 0  , 0, 0, 0, 0)
#define MMA(mt, a) do { MM1(mt, 0, a); MM1(mt, 1, a); MM1(mt, 2, a); MM1(mt, 3, a); } while (0)
#define SB() __builtin_amdgcn_sched_barrier(0)
    i32x8 bf[4], a0, a1, a2, a3, a4;
    wait_min4(lds, FULL_OFF, 1u);
    bf[0] = RD_B(0, 0); a0 = RD_A(0, 0); bf[1] = RD_B(0, 1); bf[2] = RD_B(0, 2); bf[3] = RD_B(0, 3); a1 = RD_A(0, 1);
    for (int ks = 0; ks < NKS - 1; ++ks) {
        const int sn = sb + STAGE == NSTAGE * STAGE ? 0 : sb + STAGE;
        SB();
        a2 = RD_A(sb, 2); a3 = RD_A(sb, 3);
        SB(); MMA(0, a0); SB(); a4 = RD_A(sb, 4);
        SB(); MMA(1, a1); SB(); a0 = RD_A(sb, 5);
        SB(); MMA(2, a2); SB(); a1 = RD_A(sb, 6);
        SB(); MMA(3, a3); SB(); a2 = RD_A(sb, 7);
        SB(); MMA(4, a4); SB(); a3 = RD_A(sb, 8);
        SB(); MMA(5, a0); SB(); a4 = RD_A(sb, 9);
        asm volatile("" ::: "memory");
        if (lane == 0) lds_st(lds, FREE_OFF + 4 * wn, (unsigned)(ks + 1));
        u32x4 fl = *(volatile LAS u32x4*)(lds + FULL_OFF);
        SB(); MMA(6, a1); SB();
        MMA(7, a2); SB();
        if ((unsigned)__builtin_amdgcn_readfirstlane(min(min(fl[0], fl[1]), min(fl[2], fl[3]))) < (unsigned)(ks + 2))
            wait_min4(lds, FULL_OFF, (unsigned)(ks + 2));
        asm volatile("" ::: "memory");
        a0 = RD_A(sn, 0); a1 = RD_A(sn, 1);
        SB(); MM1(8, 0, a3); if (ten) MM1(9, 0, a4); SB(); bf[0] = RD_B(sn, 0);
        SB(); MM1(8, 1, a3); if (ten) MM1(9, 1, a4); SB(); bf[1] = RD_B(sn, 1);
        SB(); MM1(8, 2, a3); if (ten) MM1(9, 2, a4); SB(); bf[2] = RD_B(sn, 2);
        SB(); MM1(8, 3, a3); if (ten) MM1(9, 3, a4); SB(); bf[3] = RD_B(sn, 3);
        sb = sn;
    }
    if ((lds_ld(lds, BAD_OFF) | lds_ld(lds, BAD_OFF + 4)) != 0u) scale = __builtin_nanf("");
    const float nsl = scale * -1.44269504088896341f;
    LAS unsigned char* tw = lds + PRIV_OFF + wn * 4096 + (lane & 15) * 256;
    LAS unsigned char* tr = lds + PRIV_OFF + wn * 4096 + g * 256;
    float* ocol = p.out + cb * BN + wn * 64 + 4 * (lane & 15);
#define EPI_MT(mt) do { \
        const int pos_ = p0 + (mt); \
        float* orow_ = ocol + (size_t)(((p.rank + POS_SO(pos_)) & 7) * MS + 16 * POS_MT(pos_) + g) * NS; \
        _Pragma("unroll") for (int nt = 0; nt < 4; ++nt) \
            *(LAS f32x4*)(tw + (((4 * nt + g) ^ (lane & 15)) << 4)) = silu4(acc[mt][nt], scale, nsl); \
        asm volatile("" ::: "memory"); \
        _Pragma("unroll") for (int i = 0; i < 4; ++i) { \
            const f32x4 v_ = *(const LAS f32x4*)(tr + i * 1024 + (((lane & 15) ^ (4 * i + g)) << 4)); \
            OUT_ST(orow_ + (size_t)(4 * i) * NS, v_); \
        } \
        asm volatile("" ::: "memory"); \
        } while (0)
#define HAND_MT(mt) do { \
        _Pragma("unroll") for (int nt = 0; nt < 4; ++nt) \
            *(LAS f32x4*)(lds + ho_off((mt) - NOWN, wn) + (lane & 15) * 256 + (((4 * nt + g) ^ (lane & 15)) << 4)) = acc[mt][nt]; \
        asm volatile("" ::: "memory"); \
        if (lane == 0) lds_st(lds, HAND_OFF + 4 * wn, (unsigned)((mt) - NOWN + 1)); \
        } while (0)
    SB();
    a2 = RD_A(sb, 5); a3 = RD_A(sb, 6);
    SB(); MMA(0, a0); SB(); a4 = RD_A(sb, 7);
    SB(); MMA(1, a1); SB(); a0 = RD_A(sb, 8);
    SB(); MMA(5, a2); SB(); a1 = RD_A(sb, 9);
    wait_min4(lds, FREE_OFF, (unsigned)(NKS - 1));
    SB(); MMA(6, a3); SB(); a2 = RD_A(sb, 2); HAND_MT(5);
    SB(); MMA(7, a4); SB(); a3 = RD_A(sb, 3); HAND_MT(6);
    SB(); MMA(8, a0); SB(); a4 = RD_A(sb, 4); HAND_MT(7);
    SB(); if (ten) MMA(9, a1); SB(); HAND_MT(8);
    SB(); MMA(2, a2); SB(); if (ten) HAND_MT(9);
    SB(); MMA(3, a3); SB(); EPI_MT(0);
    SB(); MMA(4, a4); SB(); EPI_MT(1);
    SB(); EPI_MT(2);
    SB(); EPI_MT(3);
    SB(); EPI_MT(4);
}
}

extern "C" void kernel_launch(int rank, int nranks, ncclComm_t comm,
                              hipStream_t stream,
                              void* const* d_in, const int* in_sizes, int n_in,
                              void* d_out, int out_size,
                              const int* in_dtypes, int out_dtype,
                              void* const* d_out_all,
                              void* const* d_ws_all, size_t ws_bytes) {
    if (nranks != NR || n_in != 4 || in_sizes[0] != MS * KD || in_sizes[1] != KD * NG || out_size != KD * NS ||
        ws_bytes < WS_NEED || (unsigned)rank >= (unsigned)NR) {
        fprintf(stderr, "unexpected problem geometry\n");
        return;
    }
    Params p;
    p.x = (const unsigned char*)d_in[0];
    p.w = (const unsigned char*)d_in[1];
    p.sx = (const float*)d_in[2];
    p.sw = (const float*)d_in[3];
    p.out = (float*)d_out;
    p.myws = (unsigned char*)d_ws_all[rank];
    const unsigned same = ((0x4Du >> rank) & 1u) ? 0x4Du : 0xB2u;
    int na = 0, nb = 0;
    for (int i = 0; i < 8; ++i) p.dws[i] = nullptr;
    for (int r = 0; r < NR; ++r) {
        if (r == rank) continue;
        if ((same >> r) & 1u) p.dws[na++] = (unsigned char*)d_ws_all[r];
        else p.dws[4 + nb++] = (unsigned char*)d_ws_all[r];
    }
    p.rank = rank;
    fused_kernel<<<256, 512, 0, stream>>>(p);
    HIP_CHECK(hipGetLastError());
}
```
